```python
import numpy as np

kernel_name = "gpu_ag_gemm_m4096_k4096_n16384_e5m2e5m2_dequant"


ABSMAX_RTOL = 0.05

COMPUTE_DTYPE = "float32"
EPILOGUE = "dequant"
IN_QUANT = None
OUT_QUANT = None
IN_FP8 = "e5m2e5m2"
IN_DTYPES = {"x": "e5m2", "w_mat": "e5m2",
             "scale_x": "float32", "scale_w": "float32"}
WORLD_SIZE = 8

def _snap_fp8(x, mant_bits, max_abs, min_normal):
    a = np.abs(x.astype(np.float64))
    _m, _e = np.frexp(a)
    q = np.where(a >= min_normal, np.ldexp(1.0, _e - 1 - mant_bits),
                 min_normal * 2.0 ** -mant_bits)
    snapped = np.minimum(np.round(a / q) * q, max_abs)
    return (np.sign(x) * snapped).astype(np.float32)

def setup_inputs():

    x = _snap_fp8(np.random.randn(4096, 4096).astype(np.float32), 2, 57344.0, 2.0 ** -14)
    w_mat = _snap_fp8(np.random.randn(4096, 16384).astype(np.float32), 2, 57344.0, 2.0 ** -14)
    scale_x = (0.02 * (1.0 + np.random.rand(1))).astype(np.float32)
    scale_w = (0.02 * (1.0 + np.random.rand(1))).astype(np.float32)
    return {"x": x, "w_mat": w_mat, "scale_x": scale_x, "scale_w": scale_w}

def reference(x, w_mat, scale_x, scale_w):
    acc = (x @ w_mat).astype(np.float32)
    y = (acc * (scale_x * scale_w)).astype(np.float32)
    return y
```

```cpp
#include <hip/hip_runtime.h>
#include <rccl/rccl.h>
#include <cstddef>
#include <cstdio>

#define HIP_CHECK(expr) do { hipError_t _e = (expr); if (_e != hipSuccess) { \
    fprintf(stderr, "HIP error %s at %s:%d\n", hipGetErrorString(_e), __FILE__, __LINE__); \
    return; } } while (0)

typedef unsigned int u32;
typedef __attribute__((ext_vector_type(4))) unsigned int u32x4;
typedef __attribute__((ext_vector_type(2))) int i32x2;
typedef __attribute__((ext_vector_type(4))) int i32x4;
typedef __attribute__((ext_vector_type(8))) int i32x8;
typedef __attribute__((ext_vector_type(4))) float f32x4;
#define LDS_AS __attribute__((address_space(3)))
typedef __attribute__((address_space(1))) unsigned int g_u32;

constexpr int NRANKS = 8;
constexpr int MS = 512;
constexpr int KD = 4096;
constexpr int NL = 2048;
constexpr int NSTEP = KD / 128;
constexpr int STG = 49152;
constexpr int NSTG = 3;
constexpr size_t WS_CTR = 0;
constexpr size_t WS_CLAIM = 32 << 10;
constexpr size_t WS_FLAG = 64 << 10;
constexpr size_t WS_TMO = 128 << 10;
#define LAGS(l) ((l) == 0 ? 0 : (l) == 1 ? 1 : (l) == 2 ? 3 : 7)
constexpr int NENT = NSTEP + LAGS(3);
constexpr size_t SLOTB = (size_t)NENT * 65536;
constexpr size_t WS_SLOT = 1 << 20;
constexpr size_t WS_NEED = WS_SLOT + 2 * SLOTB;
constexpr unsigned long long DEADLINE_TICKS = 50000000ull;

struct Params {
    const unsigned char* x;
    const unsigned char* w;
    const float* sx;
    const float* sw;
    float* out;
    unsigned char* myws;
    int rank;
    u32 perm;
    unsigned char* ws[NRANKS];
};

__device__ __forceinline__ __amdgpu_buffer_rsrc_t make_rsrc(const void* p, u32 bytes) {
    return __builtin_amdgcn_make_buffer_rsrc((void*)p, 0, bytes, 0x00020000);
}
#define SCHED __builtin_amdgcn_sched_barrier(0)
#define BAR __builtin_amdgcn_s_barrier()
#define WAIT_V(n) asm volatile("s_waitcnt vmcnt(" #n ")" ::: "memory")

__device__ __forceinline__ u32 wait_flags(const unsigned char* f, u32 call, g_u32* tmo, int lane, int need) {
    auto fr = make_rsrc(f, 1024);
    const unsigned long long t0 = __builtin_amdgcn_s_memrealtime();
    for (u32 n = 1;; ++n) {
        const u32x4 v = __builtin_amdgcn_raw_buffer_load_b128(fr, lane * 16, 0, 17);
        if (__all((v.x >= call || need < 1) && (v.y >= call || need < 2) && (min(v.z, v.w) >= call || need < 4))) return 0;
        if ((n & 15) == 0) {
            if (__hip_atomic_load(tmo, __ATOMIC_RELAXED, __HIP_MEMORY_SCOPE_AGENT) == call
                || __builtin_amdgcn_s_memrealtime() - t0 > DEADLINE_TICKS) {
                __hip_atomic_store(tmo, call, __ATOMIC_RELAXED, __HIP_MEMORY_SCOPE_AGENT);
                return 1;
            }
        }
        __builtin_amdgcn_s_sleep(1);
    }
}

__device__ __forceinline__ u32 wait_first(const unsigned char* f, u32 voff, u32 call, g_u32* tmo) {
    const unsigned long long t0 = __builtin_amdgcn_s_memrealtime();
    for (;;) {
        u32 a, b, c, res;
#define PLD(r) "global_load_dword " r ", %4, %5 sc0 sc1\n\t"
#define PCK(r) "s_waitcnt vmcnt(2)\n\tv_cmp_le_u32 vcc, %6, " r "\n\t" \
               "s_cmp_eq_u64 vcc, exec\n\ts_cbranch_scc1 2f\n\ts_sleep 0\n\t"
        asm volatile("s_movk_i32 %3, 64\n\ts_nop 3\n\t" PLD("%0") "s_sleep 3\n\t" PLD("%1") "s_sleep 3\n"
                     "1:\n\t" PLD("%2") PCK("%0") PLD("%0") PCK("%1") PLD("%1") PCK("%2")
                     "s_sub_u32 %3, %3, 1\n\ts_cmp_lg_u32 %3, 0\n\ts_cbranch_scc1 1b\n\t"
                     "s_mov_b32 %3, 1\n\ts_branch 3f\n"
                     "2:\n\ts_barrier\n\ts_mov_b32 %3, 0\n"
                     "3:\n\ts_waitcnt vmcnt(0)"
                     : "=&v"(a), "=&v"(b), "=&v"(c), "=&s"(res) : "v"(voff), "s"(f), "s"(call) : "memory", "vcc", "scc");
        if (res == 0) return 0;
        if (__hip_atomic_load(tmo, __ATOMIC_RELAXED, __HIP_MEMORY_SCOPE_AGENT) == call
            || __builtin_amdgcn_s_memrealtime() - t0 > DEADLINE_TICKS) {
            __hip_atomic_store(tmo, call, __ATOMIC_RELAXED, __HIP_MEMORY_SCOPE_AGENT);
            return 1;
        }
    }
}

__global__ __launch_bounds__(512, 2) void ag_gemm_kernel(Params) {
    typedef __attribute__((ext_vector_type(2))) unsigned long long u64x2;
    typedef __attribute__((ext_vector_type(4))) unsigned long long u64x4;
    typedef __attribute__((ext_vector_type(8))) unsigned long long u64x8;
    static_assert(offsetof(Params, out) == 0x20 && offsetof(Params, rank) == 0x30 && offsetof(Params, ws) == 0x38, "");
    u64x4 ka; u64x2 kb; unsigned long long kc; u64x8 kw;
    asm volatile("s_load_dwordx8 %0, %4, 0x0\n\ts_load_dwordx4 %1, %4, 0x20\n\t"
                 "s_load_dwordx2 %2, %4, 0x30\n\ts_load_dwordx16 %3, %4, 0x38"
                 : "=&s"(ka), "=&s"(kb), "=&s"(kc), "=&s"(kw) : "s"(__builtin_amdgcn_kernarg_segment_ptr()));
    const int tid = threadIdx.x, lane = tid & 63;
    const int wv = __builtin_amdgcn_readfirstlane(tid >> 6);
    const int bid = blockIdx.x;
    u32 xoff = (((bid >> 2) & 1) * 256 + (bid & 3) * 64 + (wv >> 1) * 16 + (lane & 15)) * KD
             + (bid >> 3) * 128 + (lane >> 4) * 32 + (wv & 1) * 16;
    u32 coff = WS_CTR + bid * 128 + wv * 4;
    asm volatile("" : "+v"(xoff), "+v"(coff));
    asm volatile("s_waitcnt lgkmcnt(0)" : "+s"(ka), "+s"(kb), "+s"(kc), "+s"(kw));
    Params p;
    p.x = (const unsigned char*)ka[0]; p.w = (const unsigned char*)ka[1]; p.sx = (const float*)ka[2]; p.sw = (const float*)ka[3];
    p.out = (float*)kb[0]; p.myws = (unsigned char*)kb[1]; p.rank = (int)(u32)kc; p.perm = (u32)(kc >> 32);
#pragma unroll
    for (int i = 0; i < NRANKS; ++i) p.ws[i] = (unsigned char*)kw[i];
    __shared__ __attribute__((aligned(16))) unsigned char smem[NSTG * STG + 16];
    unsigned char* myws = p.myws;

    typedef __attribute__((address_space(4))) const float c_float;
    u32x4 xv;
    u32 cprev;
    asm volatile("s_nop 4\n\tglobal_load_dwordx4 %0, %1, %2" : "=v"(xv) : "v"(xoff), "s"(p.x) : "memory");
    asm volatile("s_nop 4\n\tglobal_load_dword %0, %1, %2 sc1" : "=v"(cprev) : "v"(coff), "s"(myws) : "memory");
    u32 xcc;
    asm volatile("s_getreg_b32 %0, hwreg(HW_REG_XCC_ID, 0, 3)" : "=s"(xcc));
    u32 tile = (bid & ~7) | xcc, cold = 0;
    g_u32* const claim = (g_u32*)(myws + WS_CLAIM);
    const float sxv = *(c_float*)p.sx, swv = *(c_float*)p.sw;
    unsigned char* wsl = p.ws[0];
#pragma unroll
    for (int i = 1; i < NRANKS; ++i) wsl = lane == i ? p.ws[i] : wsl;
    g_u32* fdst = (g_u32*)(wsl + WS_FLAG + p.rank * 1024 + bid * 4);
    u32 soff = bid * 8192 + LAGS(bid & 3) * 65536 + tid * 16;
    coff = lane == 0 ? coff : ~0u;
    const unsigned char *bsrc, *arem0, *flags;
    float* otile;
#define SETUP() do { \
        const int rb_ = (tile >> 7) & 1, cb_ = (tile >> 3) & 15; \
        const u32 src = (p.perm >> (4 * (tile & 7))) & (NRANKS - 1); \
        unsigned char* srcws = p.ws[0]; \
        _Pragma("unroll") for (int i = 1; i < NRANKS; ++i) srcws = src == i ? p.ws[i] : srcws; \
        bsrc = p.w + cb_ * 128; \
        arem0 = srcws + WS_SLOT + rb_ * 32768; \
        flags = myws + WS_FLAG + src * 1024; \
        otile = p.out + (size_t)(src * MS + rb_ * 256) * NL + cb_ * 128; \
        asm volatile("" : "+s"(bsrc), "+s"(arem0), "+s"(flags), "+s"(otile)); \
    } while (0)
    SETUP();
    asm volatile("" : "+v"(fdst), "+v"(soff), "+v"(coff));
    WAIT_V(0);
    asm volatile("" : "+v"(xv), "+v"(cprev));
    const u32 call = __builtin_amdgcn_readfirstlane(cprev) + 1;
    const u32 slot = (call & 1) * (u32)SLOTB;
    __builtin_amdgcn_raw_buffer_store_b128(xv, make_rsrc(myws + WS_SLOT, 2 * (u32)SLOTB), soff, slot, 17);
    __builtin_amdgcn_raw_buffer_store_b32(call, make_rsrc(myws, WS_CLAIM), coff, 0, 16);

    volatile LDS_AS u32* s_bad = (volatile LDS_AS u32*)(smem + NSTG * STG);
    WAIT_V(0);
    BAR;
    if (wv == 1 && lane < NRANKS)
        __hip_atomic_store(fdst, call, __ATOMIC_RELAXED, __HIP_MEMORY_SCOPE_SYSTEM);

    const u32 lbase = (u32)(size_t)(LDS_AS unsigned char*)smem;
    const int bq = lane >> 3;
    const int bf = ((bq >> 1) & 3) | (((wv >> 1) & 1) << 2);
    const u32 bvo = bq * NL + (((lane & 7) ^ bf) << 4);
    auto issue_bw = [&](int e, int buf, int wv) {
        const unsigned char* g = bsrc + (size_t)e * (128 * NL) + wv * 16 * NL;
        asm volatile("s_mov_b32 m0, %3\n\ts_nop 3\n\t"
                     "global_load_lds_dwordx4 %0, %1\n\t"
                     "global_load_lds_dwordx4 %0, %2 offset:1024"
                     :: "v"(bvo), "s"(g), "s"(g + 8 * NL - 1024), "s"(lbase + buf * STG + 32768 + wv * 2048) : "memory");
    };
    auto issue_b = [&](int e, int buf) { issue_bw(e, buf, wv); };
    const unsigned char* arem;
    const u32 l16 = lane * 16;
#define ISSUE_A(N, e, l0, dst) ISSUE_AW(N, e, l0, dst, wv)
#define ISSUE_AW(N, e, l0, dst, wv) asm volatile("s_mov_b32 m0, %2\n\ts_nop 3\n\tglobal_load_lds_dwordx4 %0, %1" \
        A_MORE_##N :: "v"(l16), "s"(arem + (size_t)(e) * 65536 + (l0) * 8192 + (wv) * (N * 1024)), \
                      "s"(lbase + (dst) + (wv) * (N * 1024)) : "memory")
#define A_MORE_1
#define A_MORE_2 "\n\tglobal_load_lds_dwordx4 %0, %1 offset:1024"
#define A_MORE_3 A_MORE_2 "\n\tglobal_load_lds_dwordx4 %0, %1 offset:2048"
#define A_MORE_4 A_MORE_3 "\n\tglobal_load_lds_dwordx4 %0, %1 offset:3072"
    const int wm = wv >> 2, wn = wv & 3;
    f32x4 acc[8][2];

        const int aoff = wm * 4096 + lane * 16;
        const int g4 = lane >> 4, q = (lane & 15) >> 1, ph = lane & 1;
        const int fl = ((q >> 1) & 3) | ((g4 & 1) << 2);
        const int boff0 = 32768 + (g4 * 32 + q) * 128 + ph * 8 + (((wn * 2) ^ fl) << 4);
        const int boff1 = boff0 ^ 16;

        i32x2 wr[8][2][4];
        i32x4 xa[4][4];
#define RD_W(S, nt) do { const u32 ba = st + boff##nt; \
            asm volatile("ds_read_b64_tr_b8 %0, %1" : "=v"(wr[S][nt][0]) : "v"(ba)); \
            asm volatile("ds_read_b64_tr_b8 %0, %1 offset:1024" : "=v"(wr[S][nt][1]) : "v"(ba)); \
            asm volatile("ds_read_b64_tr_b8 %0, %1 offset:2048" : "=v"(wr[S][nt][2]) : "v"(ba)); \
            asm volatile("ds_read_b64_tr_b8 %0, %1 offset:3072" : "=v"(wr[S][nt][3]) : "v"(ba)); } while (0)
#define RD_X(r, o) do { \
            asm volatile("ds_read_b128 %0, %1 offset:%c2" : "=v"(xa[r][0]) : "v"(aa), "i"(o)); \
            asm volatile("ds_read_b128 %0, %1 offset:%c2" : "=v"(xa[r][1]) : "v"(aa), "i"((o) + 1024)); \
            asm volatile("ds_read_b128 %0, %1 offset:%c2" : "=v"(xa[r][2]) : "v"(aa), "i"((o) + 2048)); \
            asm volatile("ds_read_b128 %0, %1 offset:%c2" : "=v"(xa[r][3]) : "v"(aa), "i"((o) + 3072)); } while (0)
#define WF(S, nt) i32x8{wr[S][nt][0].x, wr[S][nt][0].y, wr[S][nt][1].x, wr[S][nt][1].y, \
                        wr[S][nt][2].x, wr[S][nt][2].y, wr[S][nt][3].x, wr[S][nt][3].y}
#define XF(r, i) i32x8{xa[r][2 * i].x, xa[r][2 * i].y, xa[r][2 * i].z, xa[r][2 * i].w, \
                       xa[r][2 * i + 1].x, xa[r][2 * i + 1].y, xa[r][2 * i + 1].z, xa[r][2 * i + 1].w}
#define MM(S, l, r, i, nt) acc[2 * (l) + i][nt] = __builtin_amdgcn_mfma_scale_f32_16x16x128_f8f6f4( \
            WF(S, nt), XF(r, i), acc[2 * (l) + i][nt], 1, 1, 0, 0, 0, 0)
#define MML(P, l, r) do { constexpr int S_ = ((P) + 8 - LAGS(l)) & 7; \
            MM(S_, l, r, 0, 0); MM(S_, l, r, 0, 1); MM(S_, l, r, 1, 0); MM(S_, l, r, 1, 1); SCHED; } while (0)
#define WAIT_W(n, S) asm volatile("s_waitcnt lgkmcnt(" #n ")" \
            : "+v"(wr[S][0][0]), "+v"(wr[S][0][1]), "+v"(wr[S][0][2]), "+v"(wr[S][0][3]), \
              "+v"(wr[S][1][0]), "+v"(wr[S][1][1]), "+v"(wr[S][1][2]), "+v"(wr[S][1][3]))
#define WAIT_X(n, r) asm volatile("s_waitcnt lgkmcnt(" #n ")" : "+v"(xa[r][0]), "+v"(xa[r][1]), "+v"(xa[r][2]), "+v"(xa[r][3]))
        const bool first = wv & 1;
#define STEP(st_, ISS, BODY) do { \
            const u32 st = lbase + (st_); \
            const u32 aa = st + aoff; \
            if (first) { ISS; } \
            SCHED; \
            BODY; \
            if (!first) { ISS; } \
            SCHED; \
        } while (0)
#define BODY4(P) do { \
            RD_X(3, 3 * 8192); RD_X(2, 2 * 8192); RD_W(P, 0); \
            WAIT_X(8, 3); MML(P, 3, 3); \
            RD_X(1, 8192); \
            WAIT_X(8, 2); MML(P, 2, 2); \
            RD_W(P, 1); RD_X(0, 0); \
            WAIT_X(8, 1); MML(P, 1, 1); \
            WAIT_W(0, P); WAIT_X(0, 0); MML(P, 0, 0); } while (0)
#define BODY4S(P, c) do { \
            RD_X(0, 0); RD_W(P, 0); RD_W(P, 1); \
            WAIT_W(0, P); WAIT_X(0, 0); \
            RD_X(1, 8192); RD_X(2, 2 * 8192); MML(P, 0, 0); \
            WAIT_X(4, 1); MML(P, 1, 1); \
            RD_X(3, 3 * 8192); \
            if (c) store_level(0); \
            WAIT_X(4, 2); MML(P, 2, 2); \
            WAIT_X(0, 3); MML(P, 3, 3); } while (0)
#define BODY3(P) do { \
            RD_X(2, 2 * 8192); RD_X(1, 8192); RD_W(P, 0); \
            WAIT_X(8, 2); MML(P, 2, 2); \
            RD_W(P, 1); RD_X(0, 0); \
            WAIT_X(12, 1); MML(P, 1, 1); \
            WAIT_W(0, P); WAIT_X(0, 0); MML(P, 0, 0); } while (0)
#define BODY2(P) do { \
            RD_X(1, 8192); RD_W(P, 0); RD_W(P, 1); \
            WAIT_X(8, 1); MML(P, 1, 1); \
            RD_X(0, 0); \
            WAIT_W(0, P); WAIT_X(0, 0); MML(P, 0, 0); } while (0)
#define BODY1(P) do { \
            RD_X(0, 0); RD_W(P, 0); RD_W(P, 1); \
            WAIT_W(0, P); WAIT_X(0, 0); MML(P, 0, 0); } while (0)
#define FULL(e, buf) do { ISSUE_A(4, e, 0, (buf) * STG); issue_b(e, buf); } while (0)
#define PART3(e, buf) do { ISSUE_A(3, e, 0, (buf) * STG); issue_b(e, buf); } while (0)
#define NEXT(buf) ((buf) == 2 ? 0 : (buf) + 1)
#define PREV(buf) ((buf) == 0 ? 2 : (buf) - 1)
    u32 bad;
    for (bool settled = false;;) {
        const int rb = (tile >> 7) & 1;
        arem = arem0 + slot;
        u32 tmo = 0;
        if (wv != 0) {
            __builtin_amdgcn_s_sleep(12);
            issue_b(0, 0);
            if (wv == 4) issue_bw(0, 0, 0);
        } else {
            tmo = wait_first(flags, (lane == 4 ? 64 : (lane & 2) * 16 + (lane & 1) * 4) + rb * 16, call, (g_u32*)(myws + WS_TMO));
        }
        if (wv != 0 || tmo) BAR;
        const bool clm = wv == 2 && !settled;
        if (wv == 0) {
            tmo |= wait_flags(flags, call, (g_u32*)(myws + WS_TMO), lane, 4);
            if (lane == 0) s_bad[0] = tmo | (tile >> 8);
        } else {
            if (rb) __builtin_amdgcn_s_sleep(16);
            ISSUE_A(1, 0, 0, 0);
            if (wv == 4) ISSUE_AW(1, 0, 0, 0, 0);
            issue_b(1, 1);
            ISSUE_A(2, 1, 0, STG);
            if (wv == 4) { issue_bw(1, 1, 0); ISSUE_AW(2, 1, 0, STG, 0); }
            issue_b(2, 2);
            ISSUE_A(2, 2, 0, 2 * STG);
            if (wv == 4) { issue_bw(2, 2, 0); ISSUE_AW(2, 2, 0, 2 * STG, 0); }
            if (clm && lane == 0)
                asm volatile("s_nop 4\n\tglobal_atomic_add %0, %1, %2, %3 sc0" : "=v"(cold) : "v"(tile * 128), "v"(1), "s"(claim) : "memory");
        }
#pragma unroll
        for (int i = 0; i < 8; ++i) acc[i][0] = acc[i][1] = f32x4{0.f, 0.f, 0.f, 0.f};
        if (wv == 4) WAIT_V(16); else if (clm) WAIT_V(9); else WAIT_V(8);
        BAR;
        STEP(0, , BODY1(0));
        if (wv == 4) WAIT_V(8); else if (clm) WAIT_V(5); else WAIT_V(4);
        BAR; STEP(STG, PART3(3, 0), BODY2(1));
        if (clm) WAIT_V(6); else WAIT_V(5);
        BAR; STEP(2 * STG, PART3(4, 1), BODY2(2));
        WAIT_V(5);
        if (clm) {
            asm volatile("" : "+v"(cold));
            u32 t = tile, tries = 0;
            for (u32 c = __builtin_amdgcn_readfirstlane(cold); c != call - 1;) {
                if (++tries > 100000) {
                    __hip_atomic_store((g_u32*)(myws + WS_TMO), call, __ATOMIC_RELAXED, __HIP_MEMORY_SCOPE_AGENT);
                    t |= 256;
                    break;
                }
                u32 r = 0;
                if (lane == 0) {
                    __hip_atomic_fetch_add(claim + t * 32, ~0u, __ATOMIC_RELAXED, __HIP_MEMORY_SCOPE_AGENT);
                    r = __hip_atomic_fetch_add(claim + ((t + 1) & 255) * 32, 1u, __ATOMIC_RELAXED, __HIP_MEMORY_SCOPE_AGENT);
                }
                t = (t + 1) & 255;
                c = __builtin_amdgcn_readfirstlane(r);
            }
            if (lane == 0) s_bad[1] = t;
            asm volatile("s_waitcnt lgkmcnt(0)" ::: "memory");
        }
        BAR;
        const u32 s0 = s_bad[0], s1 = s_bad[1];
        bad = __builtin_amdgcn_readfirstlane(s0);
        const u32 t = __builtin_amdgcn_readfirstlane(s1);
        if (settled || t == tile) break;
        tile = t;
        settled = true;
        SETUP();
        WAIT_V(0);
        BAR;
    }

    const float sy = __builtin_bit_cast(float, __builtin_amdgcn_readfirstlane(bad ? 0x7fc00000u : __builtin_bit_cast(u32, swv)));
    auto orr = make_rsrc(otile + wm * 32 * NL + wn * 32, 224u * NL * 4u);
    auto store_level = [&](int l) {
        LDS_AS unsigned char* t = (LDS_AS unsigned char*)smem + (wm ? (32 % 3) : (33 % 3)) * STG + 32768 + wn * 4096;
        int lane = __builtin_amdgcn_mbcnt_hi(-1, __builtin_amdgcn_mbcnt_lo(-1, 0));
        asm volatile("" : "+v"(lane));
        float scale = sxv;
        asm volatile("" : "+v"(scale));
        scale *= sy;
        LDS_AS unsigned char* tw = t + (lane & 15) * 128;
        const int c0 = (lane >> 4) ^ (lane & 7);
#pragma unroll
        for (int i = 0; i < 2; ++i)
#pragma unroll
            for (int nt = 0; nt < 2; ++nt)
                *(LDS_AS f32x4*)(tw + i * 2048 + ((c0 ^ (nt * 4)) << 4)) = acc[2 * l + i][nt] * scale;
        asm volatile("" ::: "memory");
        LDS_AS unsigned char* tr = t + (lane >> 3) * 128 + (((lane & 7) ^ (lane >> 3)) << 4);
        const int oo = (lane >> 3) * (NL * 4) + (lane & 7) * 16;
        f32x4 v[4];
#pragma unroll
        for (int j = 0; j < 4; ++j) v[j] = *(LDS_AS f32x4*)(tr + j * 1024);
#pragma unroll
        for (int j = 0; j < 4; ++j)
            __builtin_amdgcn_raw_buffer_store_b128(__builtin_bit_cast(u32x4, v[j]), orr, oo,
                                                   (l * 64 + j * 8) * (NL * 4), 16);
        asm volatile("" ::: "memory");
    };

    if (!bad) {
        STEP(0, PART3(5, 2), BODY3(3));
        WAIT_V(5); BAR; STEP(STG, PART3(6, 0), BODY3(4));
        WAIT_V(5); BAR; STEP(2 * STG, FULL(7, 1), BODY3(5));
        WAIT_V(6); BAR; STEP(0, FULL(8, 2), BODY3(6));
        WAIT_V(6); BAR; STEP(STG, FULL(9, 0), BODY4(7));
        int buf = 2;
        for (int e = 8; e < NSTEP; e += 8) {
            const bool last = e == NSTEP - 8;
            WAIT_V(6); BAR; STEP(buf * STG, FULL(e + 2, PREV(buf)), BODY4(0)); buf = NEXT(buf);
            WAIT_V(6); BAR; STEP(buf * STG, FULL(e + 3, PREV(buf)), BODY4(1)); buf = NEXT(buf);
            WAIT_V(6); BAR; STEP(buf * STG, FULL(e + 4, PREV(buf)), BODY4(2)); buf = NEXT(buf);
            WAIT_V(6); BAR; STEP(buf * STG, FULL(e + 5, PREV(buf)), BODY4(3)); buf = NEXT(buf);
            WAIT_V(6); BAR; STEP(buf * STG, FULL(e + 6, PREV(buf)), BODY4(4)); buf = NEXT(buf);
            WAIT_V(6); BAR; STEP(buf * STG, FULL(e + 7, PREV(buf)), BODY4(5)); buf = NEXT(buf);
            WAIT_V(6); BAR;
            STEP(buf * STG, if (last) ISSUE_A(3, 32, 1, (32 % 3) * STG); else FULL(e + 8, PREV(buf)), BODY4(6));
            buf = NEXT(buf);
            if (last) WAIT_V(3); else WAIT_V(6);
            BAR;
            STEP(buf * STG, if (last) { ISSUE_A(2, 33, 2, (33 % 3) * STG); ISSUE_A(2, 34, 2, (33 % 3) * STG + 16384); }
                            else FULL(e + 9, PREV(buf)), BODY4S(7, last));
            buf = NEXT(buf);
        }
        WAIT_V(8); BAR;
        STEP((32 % 3) * STG, { ISSUE_A(1, 35, 3, (31 % 3) * STG); ISSUE_A(1, 36, 3, (31 % 3) * STG + 8192); }, {
            RD_X(1, 0); RD_X(2, 8192); RD_X(3, 16384);
            WAIT_X(8, 1); MML(0, 1, 1); WAIT_X(4, 2); MML(0, 2, 2); store_level(1); WAIT_X(0, 3); MML(0, 3, 3); });
        if (first) WAIT_V(10); else WAIT_V(6);
        BAR;
        ISSUE_A(1, 37, 3, (31 % 3) * STG + 16384); ISSUE_A(1, 38, 3, (31 % 3) * STG + 24576);
        STEP((33 % 3) * STG, , {
            RD_X(0, 0); RD_X(1, 16384); RD_X(2, 8192);
            WAIT_X(8, 0); MML(1, 2, 0); RD_X(3, 24576);
            WAIT_X(8, 1); MML(2, 2, 1); WAIT_X(4, 2); MML(1, 3, 2); store_level(2); WAIT_X(0, 3); MML(2, 3, 3); });
        WAIT_V(4); BAR;
        STEP((31 % 3) * STG, , {
            RD_X(0, 0); RD_X(1, 8192); RD_X(2, 16384);
            WAIT_X(8, 0); MML(3, 3, 0); RD_X(3, 24576);
            WAIT_X(8, 1); MML(4, 3, 1); WAIT_X(4, 2); MML(5, 3, 2); WAIT_X(0, 3); MML(6, 3, 3); });
    } else {
        WAIT_V(0); BAR;
        store_level(0); store_level(1); store_level(2);
    }
    store_level(3);
}

extern "C" void kernel_launch(int rank, int nranks, ncclComm_t comm,
                              hipStream_t stream,
                              void* const* d_in, const int* in_sizes, int n_in,
                              void* d_out, int out_size,
                              const int* in_dtypes, int out_dtype,
                              void* const* d_out_all,
                              void* const* d_ws_all, size_t ws_bytes) {
    if (nranks != NRANKS || (unsigned)rank >= (unsigned)NRANKS || ws_bytes < WS_NEED || n_in < 4
        || in_sizes[0] != MS * KD || in_sizes[1] != KD * NL || out_size != NRANKS * MS * NL) {
        fprintf(stderr, "kernel_launch: unexpected problem shape\n");
        return;
    }
    Params p;
    p.x = (const unsigned char*)d_in[0];
    p.w = (const unsigned char*)d_in[1];
    p.sx = (const float*)d_in[2];
    p.sw = (const float*)d_in[3];
    p.out = (float*)d_out;
    for (int i = 0; i < NRANKS; ++i) p.ws[i] = (unsigned char*)d_ws_all[i];
    p.rank = rank;
    const u32 set_a = 0x4d;
    p.perm = 0;
    int lo = 0, hi = 4;
    for (int i = 0; i < NRANKS; ++i) {
        const int q = (rank + i) & 7;
        const bool same = ((set_a >> q) & 1) == ((set_a >> rank) & 1);
        p.perm |= (u32)q << (4 * (same ? lo++ : hi++));
    }
    p.myws = p.ws[rank];
    hipLaunchKernelGGL(ag_gemm_kernel, dim3(256), dim3(512), 0, stream, p);
    HIP_CHECK(hipGetLastError());
}
```
